```python
import numpy as np

kernel_name = "gpu_a2a_gemm_m2048_k2048_n2048_f32_none"


ABSMAX_RTOL = 0.001

COMPUTE_DTYPE = "float32"
EPILOGUE = "none"
IN_QUANT = None
OUT_QUANT = None
IN_DTYPES = {"x": "float32", "w_mat": "float32"}
WORLD_SIZE = 8

def setup_inputs():

    x = np.random.randn(2048, 2048).astype(np.float32)
    w_mat = (np.random.randn(2048, 2048) / np.sqrt(2048)).astype(np.float32)
    return {"x": x, "w_mat": w_mat}

def _epilogue(y):
    return y

def reference(x, w_mat):
    y = (x @ w_mat).astype(np.float32)
    return _epilogue(y)
```

```cpp
#include <hip/hip_runtime.h>
#include <rccl/rccl.h>
#include <cstdio>
#include <stdint.h>

#define POLL_SLEEP 1
#define AUX_ST 17
#define AUX_POLL 17
#define HIP_CHECK(expr) do { hipError_t _e = (expr); if (_e != hipSuccess) { \
    fprintf(stderr, "HIP error %s at %s:%d\n", hipGetErrorString(_e), __FILE__, __LINE__); \
    return; } } while (0)

typedef _Float16 half8 __attribute__((ext_vector_type(8)));
typedef _Float16 half4 __attribute__((ext_vector_type(4)));
typedef short s16x4 __attribute__((ext_vector_type(4)));
typedef float f32x16 __attribute__((ext_vector_type(16)));
typedef float f32x4 __attribute__((ext_vector_type(4)));
typedef unsigned int u32x4 __attribute__((ext_vector_type(4)));
typedef unsigned int u32x2 __attribute__((ext_vector_type(2)));

constexpr int NR = 8;
constexpr int NDIM = 2048;
constexpr int KB = 256;
constexpr int MT = 64;
constexpr int NT = 32;
constexpr int KS = KB / 16;
constexpr int NCHUNK = 4;
constexpr int NPIECE = 2 * KS;
constexpr int NSLOT = 4;

constexpr size_t CTR_OFF = 0;
constexpr size_t TMO_OFF = 4096;
constexpr size_t DATA_OFF = 65536;
constexpr size_t CHUNK_BYTES = (size_t)NPIECE * 1024;
constexpr size_t SLOT_BYTES = (size_t)NR * NCHUNK * CHUNK_BYTES;
constexpr size_t WS_NEEDED = DATA_OFF + NSLOT * SLOT_BYTES;

constexpr unsigned long long DEADLINE_TICKS = 50000000ull;
constexpr int AUX_SYS = 17;

#define STAMP(i) do {} while (0)

struct Params {
    const float* x;
    const float* w;
    float* out;
    char* ws[NR];
    int rank;
};

__device__ __forceinline__ half8 cvt8(f32x4 a, f32x4 b) {
    half8 h;
    h[0] = (_Float16)a[0]; h[1] = (_Float16)a[1]; h[2] = (_Float16)a[2]; h[3] = (_Float16)a[3];
    h[4] = (_Float16)b[0]; h[5] = (_Float16)b[1]; h[6] = (_Float16)b[2]; h[7] = (_Float16)b[3];
    return h;
}

__device__ __forceinline__ unsigned nz2(unsigned v) {
    return v == 0u ? 0x80008000u : v;
}

__device__ __forceinline__ __amdgpu_buffer_rsrc_t mkrsrc(const void* p, int bytes) {
    return __builtin_amdgcn_make_buffer_rsrc((void*)p, 0, bytes, 0x00020000);
}

__global__ __launch_bounds__(512, 2) void fused_a2a_gemm(Params p) {
    __shared__ __attribute__((aligned(16))) char lds[NR * KB * NT * 2];

    const int b = blockIdx.x;
    const int tid = threadIdx.x;
    const int lane = tid & 63;
    const int wave = __builtin_amdgcn_readfirstlane(tid >> 6);
    const int rank = p.rank;
    char* ws = p.ws[rank];

    STAMP(0);
    unsigned* ctr = (unsigned*)(ws + CTR_OFF) + b;
    unsigned* tmo = (unsigned*)(ws + TMO_OFF);
    const unsigned call =
        __hip_atomic_load(ctr, __ATOMIC_RELAXED, __HIP_MEMORY_SCOPE_AGENT) + 1u;

    const int xcd = b & 7;
    const int idx = b >> 3;
    const int m = idx & 3;
    const int n0 = xcd * 256 + (idx >> 2) * NT;
    const int lr = lane & 31;
    const int lh = lane >> 5;

    const bool sender = (wave == (b & 7));
    f32x4 sv[NCHUNK][2];
    if (sender) {
        const int d = (rank + (b & 7)) & 7;
        const int q = b >> 3;
        const float* src = p.x + (size_t)(d * 256 + (q & 1) * 32 + lr) * KB + (q >> 1) * 16 + lh * 8;
#pragma unroll
        for (int i = 0; i < NCHUNK; ++i) {
            const int mm = (rank + i) & 3;
            const f32x4* s4 = (const f32x4*)(src + (size_t)mm * MT * KB);
            sv[i][0] = s4[0];
            sv[i][1] = s4[1];
        }
    }
    STAMP(1);

    char* wl = lds + wave * (KB * NT * 2);
    {
        const float* wbase = p.w + (size_t)(wave * KB) * NDIM + n0;
        __amdgpu_buffer_rsrc_t wr = mkrsrc(wbase, (KB * NDIM - n0) * 4);
        const int voff = ((lane >> 3) * NDIM + (lane & 7) * 4) * 4;
        constexpr int NL = KB / 8;
        constexpr int INFL = 16;
        u32x4 t[INFL];
#pragma unroll
        for (int i = 0; i < (NL ? INFL : 0); ++i)
            t[i] = __builtin_amdgcn_raw_buffer_load_b128(wr, voff, i * 8 * NDIM * 4, 0);

        if (sender) {
            const int d = (rank + (b & 7)) & 7;
            const int q = b >> 3;
            char* dst = p.ws[d] + DATA_OFF + (size_t)(call & 3u) * SLOT_BYTES +
                        (size_t)rank * NCHUNK * CHUNK_BYTES + (size_t)q * 1024;
            __amdgpu_buffer_rsrc_t dr = mkrsrc(dst, NCHUNK * (int)CHUNK_BYTES);
#pragma unroll
            for (int i = 0; i < NCHUNK; ++i) {
                const int mm = (rank + i) & 3;
                u32x4 h = __builtin_bit_cast(u32x4, cvt8(sv[i][0], sv[i][1]));
                h[0] = nz2(h[0]); h[1] = nz2(h[1]); h[2] = nz2(h[2]); h[3] = nz2(h[3]);
                __builtin_amdgcn_raw_buffer_store_b128(h, dr, lane * 16, mm * (int)CHUNK_BYTES, AUX_ST);
            }
        }
        STAMP(2);

#pragma unroll
        for (int i = 0; i < NL; ++i) {
            f32x4 f = __builtin_bit_cast(f32x4, t[i % INFL]);
            half4 h;
            h[0] = (_Float16)f[0]; h[1] = (_Float16)f[1]; h[2] = (_Float16)f[2]; h[3] = (_Float16)f[3];
            *(half4*)(wl + i * 512 + lane * 8) = h;
            if (i + INFL < NL)
                t[i % INFL] = __builtin_amdgcn_raw_buffer_load_b128(wr, voff, (i + INFL) * 8 * NDIM * 4, 0);
            if ((i & 3) == 3) __builtin_amdgcn_sched_barrier(0);
        }
    }
    STAMP(3);

    u32x4 af[NPIECE];
    {
        const char* cb = ws + DATA_OFF + (size_t)(call & 3u) * SLOT_BYTES +
                         (size_t)(wave * NCHUNK + m) * CHUNK_BYTES;
        __amdgpu_buffer_rsrc_t xr = mkrsrc(cb, (int)CHUNK_BYTES);
        unsigned long long t0 = __builtin_amdgcn_s_memrealtime();
        while (true) {
            unsigned probe = __builtin_amdgcn_raw_buffer_load_b32(xr, lr * 1024 + 1020, 0, AUX_POLL);
            if (__all(probe != 0u)) {
#pragma unroll
                for (int i = 0; i < NPIECE; ++i)
                    af[i] = __builtin_amdgcn_raw_buffer_load_b128(xr, lane * 16, i * 1024, AUX_SYS);
                unsigned mn = 0xffffffffu;
#pragma unroll
                for (int i = 0; i < NPIECE; ++i)
                    mn = min(min(mn, min(af[i][0], af[i][1])), min(af[i][2], af[i][3]));
                if (__all(mn != 0u)) break;
            }
            if (__hip_atomic_load(tmo, __ATOMIC_RELAXED, __HIP_MEMORY_SCOPE_AGENT) == call ||
                __builtin_amdgcn_s_memrealtime() - t0 > DEADLINE_TICKS) {
                __hip_atomic_store(tmo, call, __ATOMIC_RELAXED, __HIP_MEMORY_SCOPE_AGENT);
                break;
            }
            __builtin_amdgcn_s_sleep(POLL_SLEEP);
        }
    }
    STAMP(4);

    f32x16 acc0 = {0}, acc1 = {0};
    {
        const int g = lane >> 4;
        const int li = lane & 15;
        const int taddr = ((g >> 1) * 8 + (li >> 2)) * 64 + ((g & 1) * 16 + (li & 3) * 4) * 2;
        typedef __attribute__((address_space(3))) s16x4* lds_s16x4_ptr;
#pragma unroll
        for (int ks = 0; ks < KS; ++ks) {
            s16x4 lo = __builtin_amdgcn_ds_read_tr16_b64_v4i16((lds_s16x4_ptr)(wl + taddr + ks * 1024));
            s16x4 hi = __builtin_amdgcn_ds_read_tr16_b64_v4i16((lds_s16x4_ptr)(wl + taddr + ks * 1024 + 256));
            half4 l4 = __builtin_bit_cast(half4, lo), h4 = __builtin_bit_cast(half4, hi);
            half8 bfr = {l4[0], l4[1], l4[2], l4[3], h4[0], h4[1], h4[2], h4[3]};
            acc0 = __builtin_amdgcn_mfma_f32_32x32x16_f16(__builtin_bit_cast(half8, af[2 * ks]), bfr, acc0, 0, 0, 0);
            acc1 = __builtin_amdgcn_mfma_f32_32x32x16_f16(__builtin_bit_cast(half8, af[2 * ks + 1]), bfr, acc1, 0, 0, 0);
        }
    }
    STAMP(5);

    {
        float* r = (float*)wl + lr;
#pragma unroll
        for (int i = 0; i < 16; ++i) {
            const int row = (i & 3) + 8 * (i >> 2) + 4 * lh;
            r[row * NT] = acc0[i];
            r[(row + 32) * NT] = acc1[i];
        }
    }
    __syncthreads();
    STAMP(6);
    {
        const int row = tid >> 3;
        const int c4 = (tid & 7) * 4;
        const float* r = (const float*)lds + row * NT + c4;
        f32x4 s = *(const f32x4*)r;
#pragma unroll
        for (int wv = 1; wv < NR; ++wv)
            s += *(const f32x4*)(r + wv * (KB * NT * 2 / 4));
        if (__hip_atomic_load(tmo, __ATOMIC_RELAXED, __HIP_MEMORY_SCOPE_AGENT) == call) {
            const float nan = __builtin_nanf("");
            s = (f32x4){nan, nan, nan, nan};
        }
        *(f32x4*)(p.out + (size_t)(m * MT + row) * NDIM + n0 + c4) = s;
    }
    {
        char* z = ws + DATA_OFF + (size_t)((call + 2u) & 3u) * SLOT_BYTES + (size_t)b * 4096;
        __amdgpu_buffer_rsrc_t zr = mkrsrc(z, 4096);
        u32x2 zero = {0u, 0u};
        __builtin_amdgcn_raw_buffer_store_b64(zero, zr, tid * 8, 0, AUX_SYS);
    }
    STAMP(7);
    if (tid == 0) __hip_atomic_store(ctr, call, __ATOMIC_RELAXED, __HIP_MEMORY_SCOPE_AGENT);
}

extern "C" void kernel_launch(int rank, int nranks, ncclComm_t comm,
                              hipStream_t stream,
                              void* const* d_in, const int* in_sizes, int n_in,
                              void* d_out, int out_size,
                              const int* in_dtypes, int out_dtype,
                              void* const* d_out_all,
                              void* const* d_ws_all, size_t ws_bytes) {
    if (nranks != NR || ws_bytes < WS_NEEDED || n_in < 2) {
        fprintf(stderr, "unexpected configuration\n");
        return;
    }
    Params p;
    p.x = (const float*)d_in[0];
    p.w = (const float*)d_in[1];
    p.out = (float*)d_out;
    for (int i = 0; i < NR; ++i) p.ws[i] = (char*)d_ws_all[i];
    p.rank = rank;
    fused_a2a_gemm<<<dim3(256), dim3(512), 0, stream>>>(p);
    HIP_CHECK(hipGetLastError());
}
```

```cpp
#include <hip/hip_runtime.h>
#include <rccl/rccl.h>
#include <cstdio>
#include <stdint.h>

#define POLL_SLEEP 1
#define AUX_ST 17
#define AUX_POLL 17
#define W_INFL 16
#define POLL_OFF ((lane & 1) * 16384 + 16380)
#define X_WIN 28
#define PROBE_MODE 0
#define HIP_CHECK(expr) do { hipError_t _e = (expr); if (_e != hipSuccess) { \
    fprintf(stderr, "HIP error %s at %s:%d\n", hipGetErrorString(_e), __FILE__, __LINE__); \
    return; } } while (0)

typedef _Float16 half8 __attribute__((ext_vector_type(8)));
typedef _Float16 half4 __attribute__((ext_vector_type(4)));
typedef short s16x4 __attribute__((ext_vector_type(4)));
typedef float f32x16 __attribute__((ext_vector_type(16)));
typedef float f32x4 __attribute__((ext_vector_type(4)));
typedef unsigned int u32x4 __attribute__((ext_vector_type(4)));
typedef unsigned int u32x2 __attribute__((ext_vector_type(2)));

constexpr int NR = 8;
constexpr int NDIM = 2048;
constexpr int KB = 256;
constexpr int MT = 64;
constexpr int NT = 32;
constexpr int KS = KB / 16;
constexpr int NCHUNK = 4;
constexpr int NPIECE = 2 * KS;
constexpr int NSLOT = 4;

constexpr size_t CTR_OFF = 0;
constexpr size_t TMO_OFF = 4096;
constexpr size_t DATA_OFF = 65536;
constexpr size_t CHUNK_BYTES = (size_t)NPIECE * 1024;
constexpr size_t SLOT_BYTES = (size_t)NR * NCHUNK * CHUNK_BYTES;
constexpr size_t WS_NEEDED = DATA_OFF + NSLOT * SLOT_BYTES;

constexpr unsigned long long DEADLINE_TICKS = 50000000ull;
constexpr int AUX_SYS = 17;

#define STAMP(i) do {} while (0)

struct Params {
    const float* x;
    const float* w;
    float* out;
    char* ws[NR];
    int rank;
};

__device__ __forceinline__ half8 cvt8(f32x4 a, f32x4 b) {
    half8 h;
    h[0] = (_Float16)a[0]; h[1] = (_Float16)a[1]; h[2] = (_Float16)a[2]; h[3] = (_Float16)a[3];
    h[4] = (_Float16)b[0]; h[5] = (_Float16)b[1]; h[6] = (_Float16)b[2]; h[7] = (_Float16)b[3];
    return h;
}

__device__ __forceinline__ unsigned nz2(unsigned v) {
    return v == 0u ? 0x80008000u : v;
}

__device__ __forceinline__ __amdgpu_buffer_rsrc_t mkrsrc(const void* p, int bytes) {
    return __builtin_amdgcn_make_buffer_rsrc((void*)p, 0, bytes, 0x00020000);
}

__global__ __launch_bounds__(576) void fused_a2a_gemm(Params p) {
    __shared__ __attribute__((aligned(16))) char lds[NR * KB * NT * 2];

    const int b = blockIdx.x;
    const int tid = threadIdx.x;
    const int lane = tid & 63;
    const int wave = __builtin_amdgcn_readfirstlane(tid >> 6);
    const int rank = p.rank;
    char* ws = p.ws[rank];

    STAMP(0);
    unsigned* ctr = (unsigned*)(ws + CTR_OFF) + b;
    unsigned* tmo = (unsigned*)(ws + TMO_OFF);

    const int xcd = b & 7;
    const int idx = b >> 3;
    const int m = idx & 3;
    const int n0 = xcd * 256 + (idx >> 2) * NT;
    const int lr = lane & 31;
    const int lh = lane >> 5;

    if (wave == NR) {
        const int d = (rank + (b & 7)) & 7;
        const int q = b >> 3;
        const float* src = p.x + (size_t)(d * 256 + (q & 1) * 32 + lr) * KB + (q >> 1) * 16 + lh * 8;
        f32x4 sv[NCHUNK][2];
#pragma unroll
        for (int i = 0; i < NCHUNK; ++i) {
            const int mm = (rank + i) & 3;
            const f32x4* s4 = (const f32x4*)(src + (size_t)mm * MT * KB);
            sv[i][0] = s4[0];
            sv[i][1] = s4[1];
        }
        const unsigned call =
            __hip_atomic_load(ctr, __ATOMIC_RELAXED, __HIP_MEMORY_SCOPE_AGENT) + 1u;
        STAMP(1);
        char* dst = p.ws[d] + DATA_OFF + (size_t)(call & 3u) * SLOT_BYTES +
                    (size_t)rank * NCHUNK * CHUNK_BYTES + (size_t)q * 1024;
        __amdgpu_buffer_rsrc_t dr = mkrsrc(dst, NCHUNK * (int)CHUNK_BYTES);
#pragma unroll
        for (int i = 0; i < NCHUNK; ++i) {
            const int mm = (rank + i) & 3;
            u32x4 h = __builtin_bit_cast(u32x4, cvt8(sv[i][0], sv[i][1]));
            h[0] = nz2(h[0]); h[1] = nz2(h[1]); h[2] = nz2(h[2]); h[3] = nz2(h[3]);
            __builtin_amdgcn_raw_buffer_store_b128(h, dr, lane * 16 + mm * (int)CHUNK_BYTES, 0, AUX_ST);
        }
        STAMP(2);
        {
            char* z = ws + DATA_OFF + (size_t)((call + 2u) & 3u) * SLOT_BYTES + (size_t)b * 4096;
            __amdgpu_buffer_rsrc_t zr = mkrsrc(z, 4096);
            u32x4 zero = {0u, 0u, 0u, 0u};
#pragma unroll
            for (int i = 0; i < 4; ++i)
                __builtin_amdgcn_raw_buffer_store_b128(zero, zr, lane * 16 + i * 1024, 0, AUX_SYS);
        }
        __syncthreads();
        if (lane == 0) __hip_atomic_store(ctr, call, __ATOMIC_RELAXED, __HIP_MEMORY_SCOPE_AGENT);
        return;
    }

    const unsigned call =
        __hip_atomic_load(ctr, __ATOMIC_RELAXED, __HIP_MEMORY_SCOPE_AGENT) + 1u;
    STAMP(1);

    char* wl = lds + wave * (KB * NT * 2);
    {
        const float* wbase = p.w + (size_t)(wave * KB) * NDIM + n0;
        __amdgpu_buffer_rsrc_t wr = mkrsrc(wbase, (KB * NDIM - n0) * 4);
        const int voff = ((lane >> 3) * NDIM + (lane & 7) * 4) * 4;
        constexpr int NL = KB / 8;
        constexpr int INFL = W_INFL;
        u32x4 t[INFL];
#pragma unroll
        for (int i = 0; i < (NL ? INFL : 0); ++i)
            t[i] = __builtin_amdgcn_raw_buffer_load_b128(wr, voff, i * 8 * NDIM * 4, 0);
        STAMP(2);
#pragma unroll
        for (int i = 0; i < NL; ++i) {
            f32x4 f = __builtin_bit_cast(f32x4, t[i % INFL]);
            half4 h;
            h[0] = (_Float16)f[0]; h[1] = (_Float16)f[1]; h[2] = (_Float16)f[2]; h[3] = (_Float16)f[3];
            *(half4*)(wl + i * 512 + lane * 8) = h;
            if (i + INFL < NL)
                t[i % INFL] = __builtin_amdgcn_raw_buffer_load_b128(wr, voff, (i + INFL) * 8 * NDIM * 4, 0);
            __builtin_amdgcn_sched_barrier(0);
        }
    }
    STAMP(3);

    f32x16 acc[2] = {{0}, {0}};
    {
        const char* cb = ws + DATA_OFF + (size_t)(call & 3u) * SLOT_BYTES +
                         (size_t)(wave * NCHUNK + m) * CHUNK_BYTES;
        __amdgpu_buffer_rsrc_t xr = mkrsrc(cb, (int)CHUNK_BYTES);
        const int g = lane >> 4;
        const int li = lane & 15;
        const int taddr = ((g >> 1) * 8 + (li >> 2)) * 64 + ((g & 1) * 16 + (li & 3) * 4) * 2;
        typedef __attribute__((address_space(3))) s16x4* lds_s16x4_ptr;
        unsigned long long t0 = __builtin_amdgcn_s_memrealtime();
        bool alive = true;
        bool wide = (PROBE_MODE == 2);
        unsigned spins = 0;
        while (alive) {
            unsigned probe = __builtin_amdgcn_raw_buffer_load_b32(
                xr, wide ? lr * 1024 + 1020 : (lane & 1) * 16384 + 16380, 0, AUX_POLL);
            if (__all(probe != 0u)) {
                if (wide || PROBE_MODE == 0) break;
                wide = true;
                continue;
            }
            if ((++spins & 31u) == 0u) {
                if (__hip_atomic_load(tmo, __ATOMIC_RELAXED, __HIP_MEMORY_SCOPE_AGENT) == call ||
                    __builtin_amdgcn_s_memrealtime() - t0 > DEADLINE_TICKS) {
                    __hip_atomic_store(tmo, call, __ATOMIC_RELAXED, __HIP_MEMORY_SCOPE_AGENT);
                    alive = false;
                }
            }
            __builtin_amdgcn_s_sleep(POLL_SLEEP);
        }
        STAMP(4);
        constexpr int XW = X_WIN;
        while (true) {
            u32x4 af[XW];
            unsigned mn = 0xffffffffu;
            acc[0] = (f32x16){0}; acc[1] = (f32x16){0};
#pragma unroll
            for (int i = 0; i < XW; ++i)
                af[i] = __builtin_amdgcn_raw_buffer_load_b128(xr, lane * 16, i * 1024, AUX_SYS);
#pragma unroll
            for (int ks = 0; ks < KS; ++ks) {
                s16x4 lo = __builtin_amdgcn_ds_read_tr16_b64_v4i16((lds_s16x4_ptr)(wl + taddr + ks * 1024));
                s16x4 hi = __builtin_amdgcn_ds_read_tr16_b64_v4i16((lds_s16x4_ptr)(wl + taddr + ks * 1024 + 256));
                half4 l4 = __builtin_bit_cast(half4, lo), h4 = __builtin_bit_cast(half4, hi);
                half8 bfr = {l4[0], l4[1], l4[2], l4[3], h4[0], h4[1], h4[2], h4[3]};
#pragma unroll
                for (int t = 0; t < 2; ++t) {
                    const int pc = 2 * ks + t;
                    const u32x4 a = af[pc % XW];
                    mn = min(min(mn, min(a[0], a[1])), min(a[2], a[3]));
                    acc[t] = __builtin_amdgcn_mfma_f32_32x32x16_f16(__builtin_bit_cast(half8, a), bfr, acc[t], 0, 0, 0);
                    if (pc + XW < NPIECE)
                        af[pc % XW] = __builtin_amdgcn_raw_buffer_load_b128(xr, lane * 16, (pc + XW) * 1024, AUX_SYS);
                }
            }
            if (__all(mn != 0u) || !alive) break;
            if (__hip_atomic_load(tmo, __ATOMIC_RELAXED, __HIP_MEMORY_SCOPE_AGENT) == call ||
                __builtin_amdgcn_s_memrealtime() - t0 > DEADLINE_TICKS) {
                __hip_atomic_store(tmo, call, __ATOMIC_RELAXED, __HIP_MEMORY_SCOPE_AGENT);
                alive = false;
            }
        }
    }
    STAMP(5);

    {
        float* r = (float*)wl + lr;
#pragma unroll
        for (int i = 0; i < 16; ++i) {
            const int row = (i & 3) + 8 * (i >> 2) + 4 * lh;
            r[row * NT] = acc[0][i];
            r[(row + 32) * NT] = acc[1][i];
        }
    }
    __syncthreads();
    STAMP(6);
    {
        const int row = tid >> 3;
        const int c4 = (tid & 7) * 4;
        const float* r = (const float*)lds + row * NT + c4;
        f32x4 s = *(const f32x4*)r;
#pragma unroll
        for (int wv = 1; wv < NR; ++wv)
            s += *(const f32x4*)(r + wv * (KB * NT * 2 / 4));
        if (__hip_atomic_load(tmo, __ATOMIC_RELAXED, __HIP_MEMORY_SCOPE_AGENT) == call) {
            const float nan = __builtin_nanf("");
            s = (f32x4){nan, nan, nan, nan};
        }
        *(f32x4*)(p.out + (size_t)(m * MT + row) * NDIM + n0 + c4) = s;
    }
    STAMP(7);
}

extern "C" void kernel_launch(int rank, int nranks, ncclComm_t comm,
                              hipStream_t stream,
                              void* const* d_in, const int* in_sizes, int n_in,
                              void* d_out, int out_size,
                              const int* in_dtypes, int out_dtype,
                              void* const* d_out_all,
                              void* const* d_ws_all, size_t ws_bytes) {
    if (nranks != NR || ws_bytes < WS_NEEDED || n_in < 2) {
        fprintf(stderr, "unexpected configuration\n");
        return;
    }
    Params p;
    p.x = (const float*)d_in[0];
    p.w = (const float*)d_in[1];
    p.out = (float*)d_out;
    for (int i = 0; i < NR; ++i) p.ws[i] = (char*)d_ws_all[i];
    p.rank = rank;
    fused_a2a_gemm<<<dim3(256), dim3(576), 0, stream>>>(p);
    HIP_CHECK(hipGetLastError());
}
```

```cpp
#include <hip/hip_runtime.h>
#include <rccl/rccl.h>
#include <cstdio>
#include <stdint.h>

#define W_ROT 8
#define AUX_X 17
#define X_ROT 0
#define EARLY_PROBE 8
#define X_TWO_PHASE 0
#define POLL_SLEEP 1
#define AUX_ST 17
#define AUX_POLL 17
#define W_INFL 4
#define POLL_OFF ((lane & 1) * 16384 + 16380)
#define X_WIN 8
#define PROBE_MODE 0
#define AUX_W 0
#define HIP_CHECK(expr) do { hipError_t _e = (expr); if (_e != hipSuccess) { \
    fprintf(stderr, "HIP error %s at %s:%d\n", hipGetErrorString(_e), __FILE__, __LINE__); \
    return; } } while (0)

typedef _Float16 half8 __attribute__((ext_vector_type(8)));
typedef _Float16 half4 __attribute__((ext_vector_type(4)));
typedef short s16x4 __attribute__((ext_vector_type(4)));
typedef float f32x16 __attribute__((ext_vector_type(16)));
typedef float f32x4 __attribute__((ext_vector_type(4)));
typedef unsigned int u32x4 __attribute__((ext_vector_type(4)));
typedef unsigned int u32x2 __attribute__((ext_vector_type(2)));

constexpr int NR = 8;
constexpr int NDIM = 2048;
constexpr int KB = 256;
constexpr int MT = 64;
constexpr int NT = 32;
constexpr int KS = KB / 16;
constexpr int NCHUNK = 4;
constexpr int NPIECE = 2 * KS;
constexpr int NSLOT = 4;

constexpr size_t CTR_OFF = 0;
constexpr size_t TMO_OFF = 4096;
constexpr size_t DATA_OFF = 65536;
constexpr size_t CHUNK_BYTES = (size_t)NPIECE * 1024;
constexpr size_t SLOT_BYTES = (size_t)NR * NCHUNK * CHUNK_BYTES;
constexpr size_t WS_NEEDED = DATA_OFF + NSLOT * SLOT_BYTES;

constexpr unsigned long long DEADLINE_TICKS = 50000000ull;
constexpr int AUX_SYS = 17;

#define STAMP(i) do {} while (0)

struct Params {
    const float* x;
    const float* w;
    float* out;
    char* ws[NR];
    int rank;
};

__device__ __forceinline__ half8 cvt8(f32x4 a, f32x4 b) {
    half8 h;
    h[0] = (_Float16)a[0]; h[1] = (_Float16)a[1]; h[2] = (_Float16)a[2]; h[3] = (_Float16)a[3];
    h[4] = (_Float16)b[0]; h[5] = (_Float16)b[1]; h[6] = (_Float16)b[2]; h[7] = (_Float16)b[3];
    return h;
}

__device__ __forceinline__ unsigned nz2(unsigned v) {
    return v == 0u ? 0x80008000u : v;
}

__device__ __forceinline__ __amdgpu_buffer_rsrc_t mkrsrc(const void* p, int bytes) {
    return __builtin_amdgcn_make_buffer_rsrc((void*)p, 0, bytes, 0x00020000);
}

__global__ __launch_bounds__(576) void fused_a2a_gemm(Params p) {
    __shared__ __attribute__((aligned(16))) char lds[NR * KB * NT * 2];

    const int b = blockIdx.x;
    const int tid = threadIdx.x;
    const int lane = tid & 63;
    const int wave = __builtin_amdgcn_readfirstlane(tid >> 6);
    const int rank = p.rank;
    char* ws = p.ws[rank];

    STAMP(0);
    unsigned* ctr = (unsigned*)(ws + CTR_OFF) + b;
    unsigned* tmo = (unsigned*)(ws + TMO_OFF);

    const int xcd = b & 7;
    const int idx = b >> 3;
    const int m = idx & 3;
    const int n0 = xcd * 256 + (idx >> 2) * NT;
    const int lr = lane & 31;
    const int lh = lane >> 5;

    if (wave == NR) {
        const int d = (rank + (b & 7)) & 7;
        const int q = b >> 3;
        const float* src = p.x + (size_t)(d * 256 + (q & 1) * 32 + lr) * KB + (q >> 1) * 16 + lh * 8;
        f32x4 sv[NCHUNK][2];
#pragma unroll
        for (int i = 0; i < NCHUNK; ++i) {
            const int mm = (rank + i) & 3;
            const f32x4* s4 = (const f32x4*)(src + (size_t)mm * MT * KB);
            sv[i][0] = s4[0];
            sv[i][1] = s4[1];
        }
        const unsigned call =
            __hip_atomic_load(ctr, __ATOMIC_RELAXED, __HIP_MEMORY_SCOPE_AGENT) + 1u;
        STAMP(1);
        char* dst = p.ws[d] + DATA_OFF + (size_t)(call & 3u) * SLOT_BYTES +
                    (size_t)rank * NCHUNK * CHUNK_BYTES + (size_t)q * 1024;
        __amdgpu_buffer_rsrc_t dr = mkrsrc(dst, NCHUNK * (int)CHUNK_BYTES);
#pragma unroll
        for (int i = 0; i < NCHUNK; ++i) {
            const int mm = (rank + i) & 3;
            u32x4 h = __builtin_bit_cast(u32x4, cvt8(sv[i][0], sv[i][1]));
            h[0] = nz2(h[0]); h[1] = nz2(h[1]); h[2] = nz2(h[2]); h[3] = nz2(h[3]);
            __builtin_amdgcn_raw_buffer_store_b128(h, dr, lane * 16 + mm * (int)CHUNK_BYTES, 0, AUX_ST);
        }
        STAMP(2);
        {
            char* z = ws + DATA_OFF + (size_t)((call + 2u) & 3u) * SLOT_BYTES + (size_t)b * 4096;
            __amdgpu_buffer_rsrc_t zr = mkrsrc(z, 4096);
            u32x4 zero = {0u, 0u, 0u, 0u};
#pragma unroll
            for (int i = 0; i < 4; ++i)
                __builtin_amdgcn_raw_buffer_store_b128(zero, zr, lane * 16 + i * 1024, 0, AUX_SYS);
        }
        __syncthreads();
        if (lane == 0) __hip_atomic_store(ctr, call, __ATOMIC_RELAXED, __HIP_MEMORY_SCOPE_AGENT);
        return;
    }

    const unsigned call =
        __hip_atomic_load(ctr, __ATOMIC_RELAXED, __HIP_MEMORY_SCOPE_AGENT) + 1u;
    STAMP(1);

    char* wl = lds + wave * (KB * NT * 2);
    const char* cb = ws + DATA_OFF + (size_t)(call & 3u) * SLOT_BYTES +
                     (size_t)(wave * NCHUNK + m) * CHUNK_BYTES;
    __amdgpu_buffer_rsrc_t xr = mkrsrc(cb, (int)CHUNK_BYTES);
    unsigned probe0 = 0u;
    {
        const float* wbase = p.w + (size_t)(wave * KB) * NDIM + n0;
        __amdgpu_buffer_rsrc_t wr = mkrsrc(wbase, (KB * NDIM - n0) * 4);
        const int voff = ((lane >> 3) * NDIM + (lane & 7) * 4) * 4;
        constexpr int NL = KB / 8;
        constexpr int INFL = W_INFL;
        u32x4 t[INFL];
        const int rot = m * W_ROT;
#pragma unroll
        for (int i = 0; i < (NL ? INFL : 0); ++i)
            t[i] = __builtin_amdgcn_raw_buffer_load_b128(wr, voff, ((i + rot) & 31) * 8 * NDIM * 4, AUX_W);
        STAMP(2);
#pragma unroll
        for (int i = 0; i < NL; ++i) {
            f32x4 f = __builtin_bit_cast(f32x4, t[i % INFL]);
            half4 h;
            h[0] = (_Float16)f[0]; h[1] = (_Float16)f[1]; h[2] = (_Float16)f[2]; h[3] = (_Float16)f[3];
            *(half4*)(wl + ((i + rot) & 31) * 512 + lane * 8) = h;
            if (i + INFL < NL)
                t[i % INFL] = __builtin_amdgcn_raw_buffer_load_b128(wr, voff, ((i + INFL + rot) & 31) * 8 * NDIM * 4, AUX_W);
            if (i == NL - EARLY_PROBE)
                probe0 = __builtin_amdgcn_raw_buffer_load_b32(xr, (lane & 1) * 16384 + 16380, 0, AUX_POLL);
            __builtin_amdgcn_sched_barrier(0);
        }
    }
    STAMP(3);

    f32x16 acc[2] = {{0}, {0}};
    {
        const int g = lane >> 4;
        const int li = lane & 15;
        const int taddr = ((g >> 1) * 8 + (li >> 2)) * 64 + ((g & 1) * 16 + (li & 3) * 4) * 2;
        typedef __attribute__((address_space(3))) s16x4* lds_s16x4_ptr;
        unsigned long long t0 = __builtin_amdgcn_s_memrealtime();
        bool alive = true;
        bool wide = (PROBE_MODE == 2);
        unsigned spins = 0;
        if (__all(probe0 != 0u)) wide = true, spins = ~0u;
        while (alive && spins != ~0u) {
            unsigned probe = __builtin_amdgcn_raw_buffer_load_b32(
                xr, wide ? lr * 1024 + 1020 : (lane & 1) * 16384 + 16380, 0, AUX_POLL);
            if (__all(probe != 0u)) {
                if (wide || PROBE_MODE == 0) break;
                wide = true;
                continue;
            }
            if ((++spins & 31u) == 0u) {
                if (__hip_atomic_load(tmo, __ATOMIC_RELAXED, __HIP_MEMORY_SCOPE_AGENT) == call ||
                    __builtin_amdgcn_s_memrealtime() - t0 > DEADLINE_TICKS) {
                    __hip_atomic_store(tmo, call, __ATOMIC_RELAXED, __HIP_MEMORY_SCOPE_AGENT);
                    alive = false;
                }
            }
            __builtin_amdgcn_s_sleep(POLL_SLEEP);
        }
        STAMP(4);
        constexpr int XW = X_WIN;
        while (true) {
            constexpr int XB = 20;
            u32x4 af[X_TWO_PHASE ? XB : XW];
            unsigned mn = 0xffffffffu;
            acc[0] = (f32x16){0}; acc[1] = (f32x16){0};
            const int xrot = ((idx >> 2) * X_ROT) & 31;
#pragma unroll
            for (int i = 0; i < XW; ++i)
                af[i] = __builtin_amdgcn_raw_buffer_load_b128(xr, lane * 16, ((i + xrot) & 31) * 1024, AUX_X);
#pragma unroll
            for (int k0 = 0; k0 < KS; ++k0) {
                const int ks = (k0 + (xrot >> 1)) & 15;
                s16x4 lo = __builtin_amdgcn_ds_read_tr16_b64_v4i16((lds_s16x4_ptr)(wl + taddr + ks * 1024));
                s16x4 hi = __builtin_amdgcn_ds_read_tr16_b64_v4i16((lds_s16x4_ptr)(wl + taddr + ks * 1024 + 256));
                half4 l4 = __builtin_bit_cast(half4, lo), h4 = __builtin_bit_cast(half4, hi);
                half8 bfr = {l4[0], l4[1], l4[2], l4[3], h4[0], h4[1], h4[2], h4[3]};
#pragma unroll
                for (int t = 0; t < 2; ++t) {
                    const int pc = 2 * k0 + t;
                    const u32x4 a = af[pc % XW];
                    mn = min(min(mn, min(a[0], a[1])), min(a[2], a[3]));
                    acc[t] = __builtin_amdgcn_mfma_f32_32x32x16_f16(__builtin_bit_cast(half8, a), bfr, acc[t], 0, 0, 0);
                    if (pc + XW < NPIECE)
                        af[pc % XW] = __builtin_amdgcn_raw_buffer_load_b128(xr, lane * 16, ((pc + XW + xrot) & 31) * 1024, AUX_X);
                }
            }
            if (__all(mn != 0u) || !alive) break;
            if (__hip_atomic_load(tmo, __ATOMIC_RELAXED, __HIP_MEMORY_SCOPE_AGENT) == call ||
                __builtin_amdgcn_s_memrealtime() - t0 > DEADLINE_TICKS) {
                __hip_atomic_store(tmo, call, __ATOMIC_RELAXED, __HIP_MEMORY_SCOPE_AGENT);
                alive = false;
            }
        }
    }
    STAMP(5);

    {
        float* r = (float*)wl + lr;
#pragma unroll
        for (int i = 0; i < 16; ++i) {
            const int row = (i & 3) + 8 * (i >> 2) + 4 * lh;
            r[row * NT] = acc[0][i];
            r[(row + 32) * NT] = acc[1][i];
        }
    }
    __syncthreads();
    STAMP(6);
    {
        const int row = tid >> 3;
        const int c4 = (tid & 7) * 4;
        const float* r = (const float*)lds + row * NT + c4;
        f32x4 s = *(const f32x4*)r;
#pragma unroll
        for (int wv = 1; wv < NR; ++wv)
            s += *(const f32x4*)(r + wv * (KB * NT * 2 / 4));
        if (__hip_atomic_load(tmo, __ATOMIC_RELAXED, __HIP_MEMORY_SCOPE_AGENT) == call) {
            const float nan = __builtin_nanf("");
            s = (f32x4){nan, nan, nan, nan};
        }
        *(f32x4*)(p.out + (size_t)(m * MT + row) * NDIM + n0 + c4) = s;
    }
    STAMP(7);
}

extern "C" void kernel_launch(int rank, int nranks, ncclComm_t comm,
                              hipStream_t stream,
                              void* const* d_in, const int* in_sizes, int n_in,
                              void* d_out, int out_size,
                              const int* in_dtypes, int out_dtype,
                              void* const* d_out_all,
                              void* const* d_ws_all, size_t ws_bytes) {
    if (nranks != NR || ws_bytes < WS_NEEDED || n_in < 2) {
        fprintf(stderr, "unexpected configuration\n");
        return;
    }
    Params p;
    p.x = (const float*)d_in[0];
    p.w = (const float*)d_in[1];
    p.out = (float*)d_out;
    for (int i = 0; i < NR; ++i) p.ws[i] = (char*)d_ws_all[i];
    p.rank = rank;
    fused_a2a_gemm<<<dim3(256), dim3(576), 0, stream>>>(p);
    HIP_CHECK(hipGetLastError());
}
```

```cpp
#include <hip/hip_runtime.h>
#include <rccl/rccl.h>
#include <cstdio>
#include <stdint.h>

#define HIP_CHECK(expr) do { hipError_t _e = (expr); if (_e != hipSuccess) { \
    fprintf(stderr, "HIP error %s at %s:%d\n", hipGetErrorString(_e), __FILE__, __LINE__); \
    return; } } while (0)

typedef _Float16 half8 __attribute__((ext_vector_type(8)));
typedef _Float16 half4 __attribute__((ext_vector_type(4)));
typedef short s16x4 __attribute__((ext_vector_type(4)));
typedef float f32x16 __attribute__((ext_vector_type(16)));
typedef float f32x4 __attribute__((ext_vector_type(4)));
typedef unsigned int u32x4 __attribute__((ext_vector_type(4)));

constexpr int NR = 8;
constexpr int NDIM = 2048;
constexpr int KB = 256;
constexpr int MT = 64;
constexpr int NT = 32;
constexpr int KS = KB / 16;
constexpr int NCHUNK = 4;
constexpr int NPIECE = 2 * KS;

constexpr int W_LOADS = KB / 8;
constexpr int W_INFL = 4;
constexpr int X_WIN = 8;
constexpr int EARLY_PROBE = 8;

constexpr size_t CTR_OFF = 0;
constexpr size_t TMO_OFF = 4096;
constexpr size_t DATA_OFF = 65536;
constexpr size_t CHUNK_BYTES = (size_t)NPIECE * 1024;
constexpr size_t SLOT_BYTES = (size_t)NR * NCHUNK * CHUNK_BYTES;
constexpr size_t WS_NEEDED = DATA_OFF + 4 * SLOT_BYTES;

constexpr unsigned long long DEADLINE_TICKS = 50000000ull;
constexpr int AUX_SYS = 17;

#define STAMP(i) do {} while (0)

struct Params {
    const float* x;
    const float* w;
    float* out;
    char* ws[NR];
    int rank;
};

__device__ __forceinline__ half8 cvt8(f32x4 a, f32x4 b) {
    half8 h;
    h[0] = (_Float16)a[0]; h[1] = (_Float16)a[1]; h[2] = (_Float16)a[2]; h[3] = (_Float16)a[3];
    h[4] = (_Float16)b[0]; h[5] = (_Float16)b[1]; h[6] = (_Float16)b[2]; h[7] = (_Float16)b[3];
    return h;
}

__device__ __forceinline__ unsigned nz2(unsigned v) {
    return v == 0u ? 0x80008000u : v;
}

__device__ __forceinline__ __amdgpu_buffer_rsrc_t mkrsrc(const void* p, int bytes) {
    return __builtin_amdgcn_make_buffer_rsrc((void*)p, 0, bytes, 0x00020000);
}

__device__ __forceinline__ bool give_up(unsigned* tmo, unsigned call, unsigned long long t0) {
    if (__hip_atomic_load(tmo, __ATOMIC_RELAXED, __HIP_MEMORY_SCOPE_AGENT) == call ||
        __builtin_amdgcn_s_memrealtime() - t0 > DEADLINE_TICKS) {
        __hip_atomic_store(tmo, call, __ATOMIC_RELAXED, __HIP_MEMORY_SCOPE_AGENT);
        return true;
    }
    return false;
}

__global__ __launch_bounds__(576) void fused_a2a_gemm(Params p) {
    __shared__ __attribute__((aligned(16))) char lds[NR * KB * NT * 2];

    const int b = blockIdx.x;
    const int tid = threadIdx.x;
    const int lane = tid & 63;
    const int wave = __builtin_amdgcn_readfirstlane(tid >> 6);
    const int rank = p.rank;
    char* ws = p.ws[rank];

    STAMP(0);
    unsigned* ctr = (unsigned*)(ws + CTR_OFF) + b;
    unsigned* tmo = (unsigned*)(ws + TMO_OFF);

    const int xcd = b & 7;
    const int idx = b >> 3;
    const int m = idx & 3;
    const int n0 = xcd * 256 + (idx >> 2) * NT;
    const int lr = lane & 31;
    const int lh = lane >> 5;

    if (wave == NR) {
        const int d = (rank + (b & 7)) & 7;
        const int q = b >> 3;
        const float* src = p.x + (size_t)(d * 256 + (q & 1) * 32 + lr) * KB + (q >> 1) * 16 + lh * 8;
        f32x4 sv[NCHUNK][2];
#pragma unroll
        for (int i = 0; i < NCHUNK; ++i) {
            const int mm = (rank + i) & 3;
            const f32x4* s4 = (const f32x4*)(src + (size_t)mm * MT * KB);
            sv[i][0] = s4[0];
            sv[i][1] = s4[1];
        }
        const unsigned call =
            __hip_atomic_load(ctr, __ATOMIC_RELAXED, __HIP_MEMORY_SCOPE_AGENT) + 1u;
        STAMP(1);
        char* dst = p.ws[d] + DATA_OFF + (size_t)(call & 3u) * SLOT_BYTES +
                    (size_t)rank * NCHUNK * CHUNK_BYTES + (size_t)q * 1024;
        __amdgpu_buffer_rsrc_t dr = mkrsrc(dst, NCHUNK * (int)CHUNK_BYTES);
#pragma unroll
        for (int i = 0; i < NCHUNK; ++i) {
            const int mm = (rank + i) & 3;
            u32x4 h = __builtin_bit_cast(u32x4, cvt8(sv[i][0], sv[i][1]));
            h[0] = nz2(h[0]); h[1] = nz2(h[1]); h[2] = nz2(h[2]); h[3] = nz2(h[3]);
            __builtin_amdgcn_raw_buffer_store_b128(h, dr, lane * 16 + mm * (int)CHUNK_BYTES, 0, AUX_SYS);
        }
        STAMP(2);
        {
            char* z = ws + DATA_OFF + (size_t)((call + 2u) & 3u) * SLOT_BYTES + (size_t)b * 4096;
            __amdgpu_buffer_rsrc_t zr = mkrsrc(z, 4096);
            u32x4 zero = {0u, 0u, 0u, 0u};
#pragma unroll
            for (int i = 0; i < 4; ++i)
                __builtin_amdgcn_raw_buffer_store_b128(zero, zr, lane * 16 + i * 1024, 0, AUX_SYS);
        }
        __syncthreads();
        if (lane == 0) __hip_atomic_store(ctr, call, __ATOMIC_RELAXED, __HIP_MEMORY_SCOPE_AGENT);
        return;
    }

    const unsigned call =
        __hip_atomic_load(ctr, __ATOMIC_RELAXED, __HIP_MEMORY_SCOPE_AGENT) + 1u;
    STAMP(1);
    char* wl = lds + wave * (KB * NT * 2);
    const float* wbase = p.w + (size_t)(wave * KB) * NDIM + n0;
    const __amdgpu_buffer_rsrc_t wr = mkrsrc(wbase, (KB * NDIM - n0) * 4);
    const int voff = ((lane >> 3) * NDIM + (lane & 7) * 4) * 4;
    const char* cb = ws + DATA_OFF + (size_t)(call & 3u) * SLOT_BYTES +
                     (size_t)(wave * NCHUNK + m) * CHUNK_BYTES;
    const __amdgpu_buffer_rsrc_t xr = mkrsrc(cb, (int)CHUNK_BYTES);
    const int poff = (lane & 1) * 16384 + 16380;
    unsigned probe = 0u;
    const int rot = m * (W_LOADS / 4);
    u32x4 t[W_INFL];
    asm volatile("" :: "s"(xr));
    __builtin_amdgcn_sched_barrier(0);
#pragma unroll
    for (int i = 0; i < W_INFL; ++i)
        t[i] = __builtin_amdgcn_raw_buffer_load_b128(wr, voff, ((i + rot) & 31) * 8 * NDIM * 4, 0);
    STAMP(2);
#pragma unroll
    for (int i = 0; i < W_LOADS; ++i) {
        f32x4 f = __builtin_bit_cast(f32x4, t[i % W_INFL]);
        half4 h;
        h[0] = (_Float16)f[0]; h[1] = (_Float16)f[1]; h[2] = (_Float16)f[2]; h[3] = (_Float16)f[3];
        *(half4*)(wl + ((i + rot) & 31) * 512 + lane * 8) = h;
        if (i + W_INFL < W_LOADS)
            t[i % W_INFL] = __builtin_amdgcn_raw_buffer_load_b128(
                wr, voff, ((i + W_INFL + rot) & 31) * 8 * NDIM * 4, 0);
        if (i == W_LOADS - EARLY_PROBE)
            probe = __builtin_amdgcn_raw_buffer_load_b32(xr, poff, 0, AUX_SYS);
        __builtin_amdgcn_sched_barrier(0);
    }
    STAMP(3);

    f32x16 acc[2];
    {
        const int g = lane >> 4;
        const int li = lane & 15;
        const int taddr = ((g >> 1) * 8 + (li >> 2)) * 64 + ((g & 1) * 16 + (li & 3) * 4) * 2;
        typedef __attribute__((address_space(3))) s16x4* lds_s16x4_ptr;
        const unsigned long long t0 = __builtin_amdgcn_s_memrealtime();
        bool alive = true;
        unsigned spins = 0;
        if (!__all(probe != 0u)) {
            while (true) {
                probe = __builtin_amdgcn_raw_buffer_load_b32(xr, poff, 0, AUX_SYS);
                if (__all(probe != 0u)) break;
                if ((++spins & 31u) == 0u && give_up(tmo, call, t0)) { alive = false; break; }
                __builtin_amdgcn_s_sleep(1);
            }
        }
        STAMP(4);
        while (true) {
            u32x4 af[X_WIN];
            unsigned mn = 0xffffffffu;
            acc[0] = (f32x16){0};
            acc[1] = (f32x16){0};
#pragma unroll
            for (int i = 0; i < X_WIN; ++i)
                af[i] = __builtin_amdgcn_raw_buffer_load_b128(xr, lane * 16, i * 1024, AUX_SYS);
#pragma unroll
            for (int ks = 0; ks < KS; ++ks) {
                s16x4 lo = __builtin_amdgcn_ds_read_tr16_b64_v4i16((lds_s16x4_ptr)(wl + taddr + ks * 1024));
                s16x4 hi = __builtin_amdgcn_ds_read_tr16_b64_v4i16((lds_s16x4_ptr)(wl + taddr + ks * 1024 + 256));
                half4 l4 = __builtin_bit_cast(half4, lo), h4 = __builtin_bit_cast(half4, hi);
                half8 bfr = {l4[0], l4[1], l4[2], l4[3], h4[0], h4[1], h4[2], h4[3]};
#pragma unroll
                for (int tt = 0; tt < 2; ++tt) {
                    const int pc = 2 * ks + tt;
                    const u32x4 a = af[pc % X_WIN];
                    mn = min(min(mn, min(a[0], a[1])), min(a[2], a[3]));
                    acc[tt] = __builtin_amdgcn_mfma_f32_32x32x16_f16(
                        __builtin_bit_cast(half8, a), bfr, acc[tt], 0, 0, 0);
                    if (pc + X_WIN < NPIECE)
                        af[pc % X_WIN] = __builtin_amdgcn_raw_buffer_load_b128(
                            xr, lane * 16, (pc + X_WIN) * 1024, AUX_SYS);
                }
            }
            if (__all(mn != 0u) || !alive) break;
            if (give_up(tmo, call, t0)) alive = false;
        }
    }
    STAMP(5);

    {
        float* r = (float*)wl + lr;
#pragma unroll
        for (int i = 0; i < 16; ++i) {
            const int row = (i & 3) + 8 * (i >> 2) + 4 * lh;
            r[row * NT] = acc[0][i];
            r[(row + 32) * NT] = acc[1][i];
        }
    }
    __syncthreads();
    STAMP(6);
    {
        const int row = tid >> 3;
        const int c4 = (tid & 7) * 4;
        const float* r = (const float*)lds + row * NT + c4;
        f32x4 s = *(const f32x4*)r;
#pragma unroll
        for (int wv = 1; wv < NR; ++wv)
            s += *(const f32x4*)(r + wv * (KB * NT * 2 / 4));
        if (__hip_atomic_load(tmo, __ATOMIC_RELAXED, __HIP_MEMORY_SCOPE_AGENT) == call) {
            const float nan = __builtin_nanf("");
            s = (f32x4){nan, nan, nan, nan};
        }
        *(f32x4*)(p.out + (size_t)(m * MT + row) * NDIM + n0 + c4) = s;
    }
    STAMP(7);
}

extern "C" void kernel_launch(int rank, int nranks, ncclComm_t comm,
                              hipStream_t stream,
                              void* const* d_in, const int* in_sizes, int n_in,
                              void* d_out, int out_size,
                              const int* in_dtypes, int out_dtype,
                              void* const* d_out_all,
                              void* const* d_ws_all, size_t ws_bytes) {
    if (nranks != NR || rank < 0 || rank >= NR || ws_bytes < WS_NEEDED || n_in < 2) {
        fprintf(stderr, "unexpected configuration\n");
        return;
    }
    Params p;
    p.x = (const float*)d_in[0];
    p.w = (const float*)d_in[1];
    p.out = (float*)d_out;
    for (int i = 0; i < NR; ++i) p.ws[i] = (char*)d_ws_all[i];
    p.rank = rank;
    fused_a2a_gemm<<<dim3(256), dim3(576), 0, stream>>>(p);
    HIP_CHECK(hipGetLastError());
}
```

```cpp
#include <hip/hip_runtime.h>
#include <rccl/rccl.h>
#include <cstdio>
#include <stdint.h>

#define HIP_CHECK(expr) do { hipError_t _e = (expr); if (_e != hipSuccess) { \
    fprintf(stderr, "HIP error %s at %s:%d\n", hipGetErrorString(_e), __FILE__, __LINE__); \
    return; } } while (0)

typedef _Float16 half8 __attribute__((ext_vector_type(8)));
typedef _Float16 half4 __attribute__((ext_vector_type(4)));
typedef short s16x4 __attribute__((ext_vector_type(4)));
typedef float f32x16 __attribute__((ext_vector_type(16)));
typedef float f32x4 __attribute__((ext_vector_type(4)));
typedef unsigned int u32x4 __attribute__((ext_vector_type(4)));

constexpr int NR = 8;
constexpr int NDIM = 2048;
constexpr int KB = 256;
constexpr int MT = 64;
constexpr int NT = 32;
constexpr int KS = KB / 16;
constexpr int NCHUNK = 4;
constexpr int NPIECE = 2 * KS;

constexpr int W_LOADS = KB / 8;
constexpr int W_INFL = 3;
constexpr int X_WIN = 8;
constexpr int EARLY_PROBE = 8;

constexpr size_t CTR_OFF = 0;
constexpr size_t TMO_OFF = 4096;
constexpr size_t DATA_OFF = 65536;
constexpr size_t CHUNK_BYTES = (size_t)NPIECE * 1024;
constexpr size_t SLOT_BYTES = (size_t)NR * NCHUNK * CHUNK_BYTES;
constexpr size_t WS_NEEDED = DATA_OFF + 4 * SLOT_BYTES;

constexpr unsigned long long DEADLINE_TICKS = 50000000ull;
constexpr int AUX_SYS = 17;

#define STAMP(i) do {} while (0)

struct Params {
    const float* x;
    const float* w;
    float* out;
    char* ws[NR];
    int rank;
};

__device__ __forceinline__ half8 cvt8(f32x4 a, f32x4 b) {
    half8 h;
    h[0] = (_Float16)a[0]; h[1] = (_Float16)a[1]; h[2] = (_Float16)a[2]; h[3] = (_Float16)a[3];
    h[4] = (_Float16)b[0]; h[5] = (_Float16)b[1]; h[6] = (_Float16)b[2]; h[7] = (_Float16)b[3];
    return h;
}

__device__ __forceinline__ unsigned nz2(unsigned v) {
    return v == 0u ? 0x80008000u : v;
}

__device__ __forceinline__ __amdgpu_buffer_rsrc_t mkrsrc(const void* p, int bytes) {
    return __builtin_amdgcn_make_buffer_rsrc((void*)p, 0, bytes, 0x00020000);
}

__device__ __forceinline__ bool give_up(unsigned* tmo, unsigned call, unsigned long long t0) {
    if (__hip_atomic_load(tmo, __ATOMIC_RELAXED, __HIP_MEMORY_SCOPE_AGENT) == call ||
        __builtin_amdgcn_s_memrealtime() - t0 > DEADLINE_TICKS) {
        __hip_atomic_store(tmo, call, __ATOMIC_RELAXED, __HIP_MEMORY_SCOPE_AGENT);
        return true;
    }
    return false;
}

__global__ __launch_bounds__(576) void fused_a2a_gemm(Params p) {
    __shared__ __attribute__((aligned(16))) char lds[NR * KB * NT * 2 + 64];

    const int b = blockIdx.x;
    const int tid = threadIdx.x;
    const int lane = tid & 63;
    const int wave = __builtin_amdgcn_readfirstlane(tid >> 6);
    const int rank = p.rank;
    char* ws = p.ws[rank];

    STAMP(0);
    unsigned* ctr = (unsigned*)(ws + CTR_OFF) + b;
    unsigned* tmo = (unsigned*)(ws + TMO_OFF);

    const int xcd = b & 7;
    const int idx = b >> 3;
    const int m = idx & 3;
    const int n0 = xcd * 256 + (idx >> 2) * NT;
    const int lr = lane & 31;
    const int lh = lane >> 5;

    if (wave == NR) {
        const int d = (rank + (b & 7)) & 7;
        const int q = b >> 3;
        const float* src = p.x + (size_t)(d * 256 + (q & 1) * 32 + lr) * KB + (q >> 1) * 16 + lh * 8;
        f32x4 sv[NCHUNK][2];
#pragma unroll
        for (int i = 0; i < NCHUNK; ++i) {
            const int mm = (rank + i) & 3;
            const f32x4* s4 = (const f32x4*)(src + (size_t)mm * MT * KB);
            sv[i][0] = s4[0];
            sv[i][1] = s4[1];
        }
        const unsigned call =
            __hip_atomic_load(ctr, __ATOMIC_RELAXED, __HIP_MEMORY_SCOPE_AGENT) + 1u;
        STAMP(1);
        char* dst = p.ws[d] + DATA_OFF + (size_t)(call & 3u) * SLOT_BYTES +
                    (size_t)rank * NCHUNK * CHUNK_BYTES + (size_t)q * 1024;
        __amdgpu_buffer_rsrc_t dr = mkrsrc(dst, NCHUNK * (int)CHUNK_BYTES);
#pragma unroll
        for (int i = 0; i < NCHUNK; ++i) {
            const int mm = (rank + i) & 3;
            u32x4 h = __builtin_bit_cast(u32x4, cvt8(sv[i][0], sv[i][1]));
            h[0] = nz2(h[0]); h[1] = nz2(h[1]); h[2] = nz2(h[2]); h[3] = nz2(h[3]);
            __builtin_amdgcn_raw_buffer_store_b128(h, dr, lane * 16 + mm * (int)CHUNK_BYTES, 0, AUX_SYS);
        }
        STAMP(2);
        {
            char* z = ws + DATA_OFF + (size_t)((call + 2u) & 3u) * SLOT_BYTES + (size_t)b * 4096;
            __amdgpu_buffer_rsrc_t zr = mkrsrc(z, 4096);
            u32x4 zero = {0u, 0u, 0u, 0u};
#pragma unroll
            for (int i = 0; i < 4; ++i)
                __builtin_amdgcn_raw_buffer_store_b128(zero, zr, lane * 16 + i * 1024, 0, AUX_SYS);
        }
        __syncthreads();
        if (lane == 0) __hip_atomic_store(ctr, call, __ATOMIC_RELAXED, __HIP_MEMORY_SCOPE_AGENT);
        return;
    }

    const unsigned call =
        __hip_atomic_load(ctr, __ATOMIC_RELAXED, __HIP_MEMORY_SCOPE_AGENT) + 1u;
    STAMP(1);
    char* wl = lds + wave * (KB * NT * 2);
    const float* wbase = p.w + (size_t)(wave * KB) * NDIM + n0;
    const __amdgpu_buffer_rsrc_t wr = mkrsrc(wbase, (KB * NDIM - n0) * 4);
    const int voff = ((lane >> 3) * NDIM + (lane & 7) * 4) * 4;
    const char* cb = ws + DATA_OFF + (size_t)(call & 3u) * SLOT_BYTES +
                     (size_t)(wave * NCHUNK + m) * CHUNK_BYTES;
    const __amdgpu_buffer_rsrc_t xr = mkrsrc(cb, (int)CHUNK_BYTES);
    const int poff = (lane & 1) * 16384 + 16380;
    unsigned probe = 0u;
    const int rot = m * (W_LOADS / 4);
    u32x4 t[W_INFL];
    asm volatile("" :: "s"(xr));
    __builtin_amdgcn_sched_barrier(0);
#pragma unroll
    for (int i = 0; i < W_INFL; ++i)
        t[i] = __builtin_amdgcn_raw_buffer_load_b128(wr, voff, ((i + rot) & 31) * 8 * NDIM * 4, 0);
    STAMP(2);
#pragma unroll
    for (int i = 0; i < W_LOADS; ++i) {
        f32x4 f = __builtin_bit_cast(f32x4, t[i % W_INFL]);
        half4 h;
        h[0] = (_Float16)f[0]; h[1] = (_Float16)f[1]; h[2] = (_Float16)f[2]; h[3] = (_Float16)f[3];
        *(half4*)(wl + ((i + rot) & 31) * 512 + lane * 8) = h;
        if (i + W_INFL < W_LOADS)
            t[i % W_INFL] = __builtin_amdgcn_raw_buffer_load_b128(
                wr, voff, ((i + W_INFL + rot) & 31) * 8 * NDIM * 4, 0);
        if (i == W_LOADS - EARLY_PROBE)
            probe = __builtin_amdgcn_raw_buffer_load_b32(xr, poff, 0, AUX_SYS);
        __builtin_amdgcn_sched_barrier(0);
    }
    STAMP(3);

    f32x16 acc[2];
    bool alive = true;
    {
        const int g = lane >> 4;
        const int li = lane & 15;
        const int taddr = ((g >> 1) * 8 + (li >> 2)) * 64 + ((g & 1) * 16 + (li & 3) * 4) * 2;
        typedef __attribute__((address_space(3))) s16x4* lds_s16x4_ptr;
        const unsigned long long t0 = __builtin_amdgcn_s_memrealtime();
        unsigned spins = 0;
        if (!__all(probe != 0u)) {
            while (true) {
                probe = __builtin_amdgcn_raw_buffer_load_b32(xr, poff, 0, AUX_SYS);
                if (__all(probe != 0u)) break;
                if ((++spins & 31u) == 0u && give_up(tmo, call, t0)) { alive = false; break; }
                __builtin_amdgcn_s_sleep(1);
            }
        }
        STAMP(4);
        while (true) {
            u32x4 af[X_WIN];
            unsigned mn = 0xffffffffu;
            acc[0] = (f32x16){0};
            acc[1] = (f32x16){0};
#pragma unroll
            for (int i = 0; i < X_WIN; ++i)
                af[i] = __builtin_amdgcn_raw_buffer_load_b128(xr, lane * 16, i * 1024, AUX_SYS);
#pragma unroll
            for (int ks = 0; ks < KS; ++ks) {
                s16x4 lo = __builtin_amdgcn_ds_read_tr16_b64_v4i16((lds_s16x4_ptr)(wl + taddr + ks * 1024));
                s16x4 hi = __builtin_amdgcn_ds_read_tr16_b64_v4i16((lds_s16x4_ptr)(wl + taddr + ks * 1024 + 256));
                half4 l4 = __builtin_bit_cast(half4, lo), h4 = __builtin_bit_cast(half4, hi);
                half8 bfr = {l4[0], l4[1], l4[2], l4[3], h4[0], h4[1], h4[2], h4[3]};
#pragma unroll
                for (int tt = 0; tt < 2; ++tt) {
                    const int pc = 2 * ks + tt;
                    const u32x4 a = af[pc % X_WIN];
                    mn = min(min(mn, min(a[0], a[1])), min(a[2], a[3]));
                    acc[tt] = __builtin_amdgcn_mfma_f32_32x32x16_f16(
                        __builtin_bit_cast(half8, a), bfr, acc[tt], 0, 0, 0);
                    if (pc + X_WIN < NPIECE)
                        af[pc % X_WIN] = __builtin_amdgcn_raw_buffer_load_b128(
                            xr, lane * 16, (pc + X_WIN) * 1024, AUX_SYS);
                }
            }
            if (__all(mn != 0u) || !alive) break;
            if (give_up(tmo, call, t0)) alive = false;
        }
    }
    STAMP(5);

    {
        float* r = (float*)wl + lr;
#pragma unroll
        for (int i = 0; i < 16; ++i) {
            const int row = (i & 3) + 8 * (i >> 2) + 4 * lh;
            r[row * NT] = acc[0][i];
            r[(row + 32) * NT] = acc[1][i];
        }
        if (lane == 0) ((unsigned*)(lds + NR * KB * NT * 2))[wave] = alive ? 0u : 1u;
    }
    __syncthreads();
    STAMP(6);
    {
        const int row = tid >> 3;
        const int c4 = (tid & 7) * 4;
        const float* r = (const float*)lds + row * NT + c4;
        f32x4 s = *(const f32x4*)r;
#pragma unroll
        for (int wv = 1; wv < NR; ++wv)
            s += *(const f32x4*)(r + wv * (KB * NT * 2 / 4));
        const u32x4 st0 = *(const u32x4*)(lds + NR * KB * NT * 2);
        const u32x4 st1 = *(const u32x4*)(lds + NR * KB * NT * 2 + 16);
        if ((st0[0] | st0[1] | st0[2] | st0[3] | st1[0] | st1[1] | st1[2] | st1[3]) != 0u) {
            const float nan = __builtin_nanf("");
            s = (f32x4){nan, nan, nan, nan};
        }
        *(f32x4*)(p.out + (size_t)(m * MT + row) * NDIM + n0 + c4) = s;
    }
    STAMP(7);
}

extern "C" void kernel_launch(int rank, int nranks, ncclComm_t comm,
                              hipStream_t stream,
                              void* const* d_in, const int* in_sizes, int n_in,
                              void* d_out, int out_size,
                              const int* in_dtypes, int out_dtype,
                              void* const* d_out_all,
                              void* const* d_ws_all, size_t ws_bytes) {
    if (nranks != NR || rank < 0 || rank >= NR || ws_bytes < WS_NEEDED || n_in < 2) {
        fprintf(stderr, "unexpected configuration\n");
        return;
    }
    Params p;
    p.x = (const float*)d_in[0];
    p.w = (const float*)d_in[1];
    p.out = (float*)d_out;
    for (int i = 0; i < NR; ++i) p.ws[i] = (char*)d_ws_all[i];
    p.rank = rank;
    fused_a2a_gemm<<<dim3(256), dim3(576), 0, stream>>>(p);
    HIP_CHECK(hipGetLastError());
}
```
